```python
import numpy as np

kernel_name = "gpu_gemm_rs_m4096_k4096_n32768_f32_silu"


ABSMAX_RTOL = 0.001

COMPUTE_DTYPE = "float32"
EPILOGUE = "silu"
IN_QUANT = None
OUT_QUANT = None
IN_DTYPES = {"x": "float32", "w_mat": "float32"}
WORLD_SIZE = 8

def setup_inputs():

    x = np.random.randn(4096, 4096).astype(np.float32)
    w_mat = (np.random.randn(4096, 32768) / np.sqrt(4096)).astype(np.float32)
    return {"x": x, "w_mat": w_mat}

def _epilogue(y):
    return (y / (1.0 + np.exp(-np.clip(y, -60.0, 60.0)))).astype(np.float32)

def reference(x, w_mat):
    y = (x @ w_mat).astype(np.float32)
    return _epilogue(y)
```

```cpp
#include <hip/hip_runtime.h>
#include <rccl/rccl.h>
#include <cstdio>
#include <cstdint>

#define HIP_CHECK(expr) do { hipError_t _e = (expr); if (_e != hipSuccess) { \
    fprintf(stderr, "HIP error %s at %s:%d\n", hipGetErrorString(_e), __FILE__, __LINE__); \
    return; } } while (0)

typedef _Float16 half8 __attribute__((ext_vector_type(8)));
typedef float f32x4 __attribute__((ext_vector_type(4)));
typedef unsigned u32x4 __attribute__((ext_vector_type(4)));

constexpr int NR = 8;
constexpr int MROWS = 4096;
constexpr int KS = 512;
constexpr int NG = 32768;
constexpr int NL = 4096;
constexpr int OUT_ROWS = 512;
constexpr int NWG = 256;
constexpr int NTHR = 512;
constexpr int G = 64;
constexpr int SUBS = G / NR;
constexpr int NGRP = 8;
constexpr size_t TILE_B = 32768;
constexpr size_t MAT_B = 16ull * 64 * TILE_B;
constexpr size_t STG_OFF = 1ull << 20;
constexpr size_t WS_NEED = STG_OFF + 4 * MAT_B;
constexpr int F_TICKET = 0, F_FIN = 32, F_TMO = 64, F_DONE = 96, F_CNT = 128;
constexpr unsigned long long DEADLINE_TICKS = 50000000ull;
constexpr int SHM_B = 131072;

struct Args {
    const float* x;
    const float* w;
    float* out[NR];
    char* ws[NR];
    int rank;
};

__device__ __forceinline__ int lds_byte(int r, int c) {
    int st = (r >> 4) * 2 + (c >> 5), rr = r & 15, cc = c & 31, ob = rr * 64 + cc * 2;
    return st * 1024 + (ob ^ (((ob >> 9) & 1) << 5));
}
__device__ __forceinline__ void stage_rc(int b, int& R, int& C) {
    int st = b / 1024, sb = b % 1024, swz = sb ^ (((sb >> 9) & 1) << 5);
    R = (st >> 1) * 16 + swz / 64; C = (st & 1) * 32 + (swz % 64) / 2;
}

__device__ __forceinline__ bool wait_ge(unsigned* flag, unsigned target, unsigned* tmo, unsigned call) {
    unsigned long long t0 = __builtin_amdgcn_s_memrealtime();
    bool ok = true;
    while (__hip_atomic_load(flag, __ATOMIC_RELAXED, __HIP_MEMORY_SCOPE_SYSTEM) < target) {
        if (__hip_atomic_load(tmo, __ATOMIC_RELAXED, __HIP_MEMORY_SCOPE_AGENT) == call ||
            __builtin_amdgcn_s_memrealtime() - t0 > DEADLINE_TICKS) {
            __hip_atomic_store(tmo, call, __ATOMIC_RELAXED, __HIP_MEMORY_SCOPE_AGENT);
            ok = false;
            break;
        }
        __builtin_amdgcn_s_sleep(2);
    }
    __builtin_amdgcn_fence(__ATOMIC_ACQUIRE, "");
    return ok;
}

__device__ __forceinline__ unsigned pack2(float a, float b) {
    typedef _Float16 h2 __attribute__((ext_vector_type(2)));
    h2 v; v.x = (_Float16)a; v.y = (_Float16)b;
    return __builtin_bit_cast(unsigned, v);
}

__global__ __launch_bounds__(NTHR, 2) void gemm_rs_silu(Args a) {
    extern __shared__ __attribute__((aligned(16))) char shm[];
    const int tid = threadIdx.x;
    unsigned* myf = reinterpret_cast<unsigned*>(a.ws[a.rank]);

    if (tid == 0) {
        unsigned t = __hip_atomic_fetch_add(myf + F_TICKET, 1u, __ATOMIC_RELAXED, __HIP_MEMORY_SCOPE_AGENT);
        *reinterpret_cast<unsigned*>(shm) = t / NWG + 1;
    }
    __syncthreads();
    const unsigned call = __builtin_amdgcn_readfirstlane(*reinterpret_cast<volatile unsigned*>(shm));
    __syncthreads();
    const size_t stg = STG_OFF + (size_t)(call & 1) * 2 * MAT_B;

    if (blockIdx.x < G) {
        const int p = (a.rank + (int)blockIdx.x) & (NR - 1);
        const int sub = blockIdx.x >> 3;
        char* dA = a.ws[p] + stg;
        char* dB = dA + MAT_B;
        unsigned* pcnt = reinterpret_cast<unsigned*>(a.ws[p]) + F_CNT;
        int R[4], C[4];
        #pragma unroll
        for (int i = 0; i < 4; ++i) { int cb = tid * 16 + i * 8192; stage_rc(cb & 16383, R[i], C[i]); R[i] += (cb >> 14) * 128; }
        const int n4 = tid & 63, kg = tid >> 6;
        for (int j = 0; j < NGRP; ++j) {
            const int kt = j * 8 + a.rank;
            for (int it = 0; it < 16 / SUBS; ++it) {
                const int mt = sub + it * SUBS;
                const float* src = a.x + (size_t)(mt * 256) * KS + j * 64;
                char* dst = dA + (size_t)(mt * 64 + kt) * TILE_B + tid * 16;
                f32x4 v[4][2];
                #pragma unroll
                for (int i = 0; i < 4; ++i) {
                    const f32x4* s = reinterpret_cast<const f32x4*>(src + (size_t)R[i] * KS + C[i]);
                    v[i][0] = s[0]; v[i][1] = s[1];
                }
                #pragma unroll
                for (int i = 0; i < 4; ++i) {
                    u32x4 o;
                    o.x = pack2(v[i][0].x, v[i][0].y); o.y = pack2(v[i][0].z, v[i][0].w);
                    o.z = pack2(v[i][1].x, v[i][1].y); o.w = pack2(v[i][1].z, v[i][1].w);
                    *reinterpret_cast<u32x4*>(dst + i * 8192) = o;
                }
            }
            for (int it = 0; it < 16 / SUBS; ++it) {
                const int nt = sub + it * SUBS;
                const float* src = a.w + (size_t)(j * 64 + kg * 8) * NG + p * NL + nt * 256 + n4 * 4;
                f32x4 v[8];
                #pragma unroll
                for (int k = 0; k < 8; ++k) v[k] = *reinterpret_cast<const f32x4*>(src + (size_t)k * NG);
                __syncthreads();
                #pragma unroll
                for (int i = 0; i < 4; ++i) {
                    u32x4 o;
                    o.x = pack2(v[0][i], v[1][i]); o.y = pack2(v[2][i], v[3][i]);
                    o.z = pack2(v[4][i], v[5][i]); o.w = pack2(v[6][i], v[7][i]);
                    const int n = n4 * 4 + i;
                    *reinterpret_cast<u32x4*>(shm + (n >> 7) * 16384 + lds_byte(n & 127, kg * 8)) = o;
                }
                __syncthreads();
                char* dst = dB + (size_t)(nt * 64 + kt) * TILE_B + tid * 16;
                #pragma unroll
                for (int i = 0; i < 4; ++i)
                    *reinterpret_cast<u32x4*>(dst + i * 8192) = *reinterpret_cast<const u32x4*>(shm + tid * 16 + i * 8192);
            }
            asm volatile("s_waitcnt vmcnt(0)" ::: "memory");
            __syncthreads();
            if (tid == 0) {
                __builtin_amdgcn_fence(__ATOMIC_RELEASE, "");
                asm volatile("s_waitcnt vmcnt(0)" ::: "memory");
                __hip_atomic_fetch_add(pcnt + 32 * j, 1u, __ATOMIC_RELAXED, __HIP_MEMORY_SCOPE_SYSTEM);
            }
        }
    }

    #define SA(b,h) (shm + ((b) * 2 + (h)) * 16384)
    #define SB(b,h) (shm + (4 + (b) * 2 + (h)) * 16384)
    #define STAGE(P, BASE, h, kt_) do { const char* _g = (BASE) + (size_t)(kt_) * TILE_B + (h) * 16384 + tid * 16; \
        for (int _i = 0; _i < 2; ++_i) \
            __builtin_amdgcn_global_load_lds((const unsigned*)(_g + _i * 8192), (unsigned*)((P) + tid * 16 + _i * 8192), 16, 0, 0); } while (0)
    #define LDA(dst,b,h) for (int m = 0; m < 4; ++m) for (int k = 0; k < 2; ++k) \
        dst[m][k] = *reinterpret_cast<const half8*>(SA(b,h) + lds_byte(wr * 64 + m * 16 + fr, k * 32 + fq * 8))
    #define LDB(dst,b,h) for (int n = 0; n < 2; ++n) for (int k = 0; k < 2; ++k) \
        dst[n][k] = *reinterpret_cast<const half8*>(SB(b,h) + lds_byte(wc * 32 + n * 16 + fr, k * 32 + fq * 8))
    #define MMA(ai,bj,At,Bt) do { __builtin_amdgcn_s_setprio(1); \
        for (int m = 0; m < 4; ++m) for (int n = 0; n < 2; ++n) for (int k = 0; k < 2; ++k) \
            acc[ai][bj][m][n] = __builtin_amdgcn_mfma_f32_16x16x32_f16(Bt[n][k], At[m][k], acc[ai][bj][m][n], 0, 0, 0); \
        __builtin_amdgcn_s_setprio(0); } while (0)
    #define WAIT_V(n) asm volatile("s_waitcnt vmcnt(" #n ")" ::: "memory")
    #define WAIT_L(n) asm volatile("s_waitcnt lgkmcnt(" #n ")" ::: "memory")
    #define BAR __builtin_amdgcn_s_barrier()
    #define SCHED __builtin_amdgcn_sched_barrier(0)

    int wgid = blockIdx.x;
    wgid = (wgid & 7) * (NWG / 8) + (wgid >> 3);
    const int gid = wgid / 128, pm = gid * 8 + (wgid % 128) % 8, pn = (wgid % 128) / 8;
    const char* gA = a.ws[a.rank] + stg + (size_t)pm * 64 * TILE_B;
    const char* gB = a.ws[a.rank] + stg + MAT_B + (size_t)pn * 64 * TILE_B;
    const int wid = tid >> 6, lane = tid & 63, wr = wid >> 2, wc = wid & 3, fr = lane & 15, fq = lane >> 4;
    f32x4 acc[2][2][4][2] = {};
    half8 At[4][2], B0[2][2], B1[2][2];
    bool ok = true;

    for (int g = 0; g < NGRP; ++g) {
        if (tid == 0) {
            bool o = wait_ge(myf + F_CNT + 32 * g, call * G, myf + F_TMO, call);
            asm volatile("s_waitcnt vmcnt(0)" ::: "memory");
            *reinterpret_cast<volatile int*>(shm) = o ? 1 : 0;
        }
        __syncthreads();
        ok = ok && (*reinterpret_cast<volatile int*>(shm) != 0);
        __syncthreads();
        const int k0 = g * 8;
        STAGE(SB(0,0), gB, 0, k0); STAGE(SA(0,0), gA, 0, k0);
        STAGE(SB(0,1), gB, 1, k0); STAGE(SA(0,1), gA, 1, k0);
        if (wr == 1) BAR;
        WAIT_V(4); BAR;
        STAGE(SB(1,0), gB, 0, k0 + 1); STAGE(SA(1,0), gA, 0, k0 + 1); STAGE(SB(1,1), gB, 1, k0 + 1);
        WAIT_V(6); BAR;
        for (int t = k0; t < k0 + 6; t += 2) {
            LDB(B0,0,0); SCHED; LDA(At,0,0); STAGE(SA(1,1), gA, 1, t + 1);
            WAIT_L(8); BAR; WAIT_L(0); MMA(0,0,At,B0); BAR; SCHED;
            LDB(B1,0,1); STAGE(SB(0,0), gB, 0, t + 2);
            BAR; WAIT_L(0); MMA(0,1,At,B1); BAR;
            LDA(At,0,1); STAGE(SA(0,0), gA, 0, t + 2);
            BAR; WAIT_L(0); MMA(1,0,At,B0); BAR; SCHED;
            STAGE(SB(0,1), gB, 1, t + 2);
            WAIT_V(6); BAR; MMA(1,1,At,B1); BAR;
            LDB(B0,1,0); SCHED; LDA(At,1,0); STAGE(SA(0,1), gA, 1, t + 2);
            WAIT_L(8); BAR; WAIT_L(0); MMA(0,0,At,B0); BAR; SCHED;
            LDB(B1,1,1); STAGE(SB(1,0), gB, 0, t + 3);
            BAR; WAIT_L(0); MMA(0,1,At,B1); BAR;
            LDA(At,1,1); STAGE(SA(1,0), gA, 0, t + 3);
            BAR; WAIT_L(0); MMA(1,0,At,B0); BAR; SCHED;
            STAGE(SB(1,1), gB, 1, t + 3);
            WAIT_V(6); BAR; MMA(1,1,At,B1); BAR;
        }
        { LDB(B0,0,0); WAIT_V(0); LDA(At,0,0); STAGE(SA(1,1), gA, 1, k0 + 7);
          BAR; WAIT_L(0); MMA(0,0,At,B0); BAR;
          LDB(B1,0,1); BAR; WAIT_L(0); MMA(0,1,At,B1); BAR;
          LDA(At,0,1); WAIT_V(4); BAR; WAIT_L(0); MMA(1,0,At,B0); MMA(1,1,At,B1); BAR; }
        { LDB(B0,1,0); LDA(At,1,0); WAIT_V(2); BAR; WAIT_L(0); MMA(0,0,At,B0); BAR;
          LDB(B1,1,1); WAIT_V(0); BAR; WAIT_L(0); MMA(0,1,At,B1); BAR;
          LDA(At,1,1); BAR; WAIT_L(0); MMA(1,0,At,B0); MMA(1,1,At,B1); BAR; }
        if (wr == 0) BAR;
    }

    {
        const int brow = pm * 256, q = brow / OUT_ROWS;
        float* o = a.out[q] + (size_t)(brow - q * OUT_ROWS) * NG + a.rank * NL + pn * 256;
        const float nanv = __builtin_nanf("");
        #pragma unroll
        for (int ai = 0; ai < 2; ++ai)
        #pragma unroll
        for (int bj = 0; bj < 2; ++bj)
        #pragma unroll
        for (int m = 0; m < 4; ++m)
        #pragma unroll
        for (int n = 0; n < 2; ++n) {
                f32x4 v = acc[ai][bj][m][n], r;
                #pragma unroll
                for (int e = 0; e < 4; ++e) {
                    float y = v[e];
                    float c = fminf(fmaxf(y, -60.0f), 60.0f);
                    r[e] = ok ? y / (1.0f + __expf(-c)) : nanv;
                }
                *reinterpret_cast<f32x4*>(o + (size_t)(ai * 128 + wr * 64 + m * 16 + fr) * NG + bj * 128 + wc * 32 + n * 16 + fq * 4) = r;
            }
        asm volatile("s_waitcnt vmcnt(0)" ::: "memory");
        __syncthreads();
        if (tid == 0) {
            __builtin_amdgcn_fence(__ATOMIC_RELEASE, "");
            asm volatile("s_waitcnt vmcnt(0)" ::: "memory");
            __hip_atomic_fetch_add(reinterpret_cast<unsigned*>(a.ws[q]) + F_DONE, 1u, __ATOMIC_RELAXED, __HIP_MEMORY_SCOPE_SYSTEM);
            unsigned f = __hip_atomic_fetch_add(myf + F_FIN, 1u, __ATOMIC_RELAXED, __HIP_MEMORY_SCOPE_AGENT);
            if (f % NWG == NWG - 1) {
                if (!wait_ge(myf + F_DONE, call * NWG, myf + F_TMO, call))
                    a.out[a.rank][(size_t)a.rank * NL] = nanv;
            }
        }
    }
}

extern "C" void kernel_launch(int rank, int nranks, ncclComm_t comm,
                              hipStream_t stream,
                              void* const* d_in, const int* in_sizes, int n_in,
                              void* d_out, int out_size,
                              const int* in_dtypes, int out_dtype,
                              void* const* d_out_all,
                              void* const* d_ws_all, size_t ws_bytes) {
    if (nranks != NR || n_in < 2 || ws_bytes < WS_NEED || out_size != OUT_ROWS * NG ||
        in_sizes[0] != MROWS * KS || in_sizes[1] != KS * NG) {
        fprintf(stderr, "gemm_rs: unexpected problem layout\n");
        return;
    }
    static bool attr_set = false;
    if (!attr_set) {
        HIP_CHECK(hipFuncSetAttribute((const void*)gemm_rs_silu, hipFuncAttributeMaxDynamicSharedMemorySize, SHM_B));
        attr_set = true;
    }
    Args a;
    a.x = (const float*)d_in[0];
    a.w = (const float*)d_in[1];
    for (int p = 0; p < NR; ++p) { a.out[p] = (float*)d_out_all[p]; a.ws[p] = (char*)d_ws_all[p]; }
    a.rank = rank;
    hipLaunchKernelGGL(gemm_rs_silu, dim3(NWG), dim3(NTHR), SHM_B, stream, a);
    HIP_CHECK(hipGetLastError());
}
```

```cpp
#include <hip/hip_runtime.h>
#include <rccl/rccl.h>
#include <cstdio>
#include <cstdint>

#define HIP_CHECK(expr) do { hipError_t _e = (expr); if (_e != hipSuccess) { \
    fprintf(stderr, "HIP error %s at %s:%d\n", hipGetErrorString(_e), __FILE__, __LINE__); \
    return; } } while (0)

typedef _Float16 half8 __attribute__((ext_vector_type(8)));
typedef float f32x4 __attribute__((ext_vector_type(4)));
typedef unsigned u32x4 __attribute__((ext_vector_type(4)));

constexpr int NR = 8;
constexpr int MROWS = 4096;
constexpr int KS = 512;
constexpr int NG = 32768;
constexpr int NL = 4096;
constexpr int OUT_ROWS = 512;
constexpr int NWG = 256;
constexpr int NTHR = 512;
#define CPXV 4
constexpr int CPX = CPXV;
constexpr int BPG = 16 / CPX;
constexpr int NCONV = NWG - 7 * CPX;
constexpr int NGRP = 8;
constexpr size_t TILE_B = 32768;
constexpr size_t MAT_B = 16ull * 64 * TILE_B;
constexpr size_t STG_OFF = 1ull << 20;
constexpr size_t OBOX_OFF = STG_OFF + 4 * MAT_B;
constexpr size_t OTIL_OFF = OBOX_OFF + MAT_B;
constexpr size_t OTIL_B = 256 * 256 * 4;
constexpr size_t WS_NEED = OTIL_OFF + 7 * CPX * OTIL_B;
constexpr int F_TICKET = 0, F_FIN = 32, F_TMO = 64, F_DONE = 96, F_CNT = 128, F_CONV = 512,
              F_OTIL = 1024;
constexpr unsigned long long DEADLINE_TICKS = 50000000ull;
constexpr int SHM_B = 131072;
#define POLL_SLEEP 16

#define STAMP(i) do {} while (0)

struct Args {
    const float* x;
    const float* w;
    float* out[NR];
    char* ws[NR];
    int rank;
};

__device__ __forceinline__ int lds_byte(int r, int c) {
    int st = (r >> 4) * 2 + (c >> 5), rr = r & 15, cc = c & 31, ob = rr * 64 + cc * 2;
    return st * 1024 + (ob ^ (((ob >> 9) & 1) << 5));
}
__device__ __forceinline__ void stage_rc(int b, int& R, int& C) {
    int st = b / 1024, sb = b % 1024, swz = sb ^ (((sb >> 9) & 1) << 5);
    R = (st >> 1) * 16 + swz / 64; C = (st & 1) * 32 + (swz % 64) / 2;
}

__device__ __forceinline__ bool wait_ge(unsigned* flag, unsigned target, unsigned* tmo, unsigned call, bool acq) {
    unsigned long long t0 = __builtin_amdgcn_s_memrealtime();
    bool ok = true;
    while (__hip_atomic_load(flag, __ATOMIC_RELAXED, __HIP_MEMORY_SCOPE_SYSTEM) < target) {
        if (__hip_atomic_load(tmo, __ATOMIC_RELAXED, __HIP_MEMORY_SCOPE_AGENT) == call ||
            __builtin_amdgcn_s_memrealtime() - t0 > DEADLINE_TICKS) {
            __hip_atomic_store(tmo, call, __ATOMIC_RELAXED, __HIP_MEMORY_SCOPE_AGENT);
            ok = false;
            break;
        }
        __builtin_amdgcn_s_sleep(POLL_SLEEP);
    }
    if (acq) __builtin_amdgcn_fence(__ATOMIC_ACQUIRE, "");
    else asm volatile("" ::: "memory");
    return ok;
}

constexpr int AUX_AGENT = 16, AUX_SYSTEM = 17;
__device__ __forceinline__ __amdgpu_buffer_rsrc_t mk_rsrc(const void* p, unsigned bytes) {
    const unsigned long long u = reinterpret_cast<unsigned long long>(p);
    const unsigned lo = __builtin_amdgcn_readfirstlane((unsigned)u), hi = __builtin_amdgcn_readfirstlane((unsigned)(u >> 32));
    return __builtin_amdgcn_make_buffer_rsrc(reinterpret_cast<void*>(((unsigned long long)hi << 32) | lo), 0, bytes, 0x00020000);
}

__device__ __forceinline__ unsigned pack2(float a, float b) {
    typedef _Float16 h2 __attribute__((ext_vector_type(2)));
    h2 v; v.x = (_Float16)a; v.y = (_Float16)b;
    return __builtin_bit_cast(unsigned, v);
}

__global__ __launch_bounds__(NTHR, 2) void gemm_rs_silu(Args a) {
    extern __shared__ __attribute__((aligned(16))) char shm[];
    const int tid = threadIdx.x;
    unsigned* myf = reinterpret_cast<unsigned*>(a.ws[a.rank]);

    STAMP(0);
    if (tid == 0) {
        unsigned t = __hip_atomic_fetch_add(myf + F_TICKET, 1u, __ATOMIC_RELAXED, __HIP_MEMORY_SCOPE_AGENT);
        *reinterpret_cast<unsigned*>(shm) = t / NWG + 1;
    }
    __syncthreads();
    const unsigned call = __builtin_amdgcn_readfirstlane(*reinterpret_cast<volatile unsigned*>(shm));
    __syncthreads();
    const size_t stg = STG_OFF + (size_t)(call & 1) * 2 * MAT_B;

    const int xg = blockIdx.x & 7, xi = blockIdx.x >> 3;
    const bool is_comm = (xg != 0) && (xi < CPX);
    const int lane = tid & 63, wid = tid >> 6;
    char* const mys = a.ws[a.rank] + stg;
    char* const obox = a.ws[a.rank] + OBOX_OFF;
    unsigned* const wcnt = reinterpret_cast<unsigned*>(shm + 65536);
    if (tid < NGRP) wcnt[tid] = 0;
    __syncthreads();
    if (!is_comm) {
        const int ci = (xg == 0) ? xi : 32 + (xg - 1) * (32 - CPX) + (xi - CPX);
        char* wl = shm + wid * 8192;
        const __amdgpu_buffer_rsrc_t myr = mk_rsrc(a.ws[a.rank], (unsigned)WS_NEED);
        const int n4 = lane & 15, kq = lane >> 4;
        for (int blk = ci; blk < 72 * NGRP; blk += NCONV) {
            const int j = blk / 72, e = (blk % 72) * 8 + wid;
            const int kt = j * 8 + a.rank;
            f32x4 d[16];
            if (e < 64) {
                const float* src = a.x + (size_t)(e * 64) * KS + j * 64;
                #pragma unroll
                for (int i = 0; i < 8; ++i) {
                    int R, C; stage_rc(i * 1024 + lane * 16, R, C);
                    const f32x4* q = reinterpret_cast<const f32x4*>(src + R * KS + C);
                    d[2 * i] = q[0]; d[2 * i + 1] = q[1];
                }
                const unsigned dof = (unsigned)(stg + (size_t)(kt * 16 + (e >> 2)) * TILE_B) + (e & 3) * 8192 + lane * 16;
                #pragma unroll
                for (int i = 0; i < 8; ++i) {
                    u32x4 o;
                    o.x = pack2(d[2 * i].x, d[2 * i].y); o.y = pack2(d[2 * i].z, d[2 * i].w);
                    o.z = pack2(d[2 * i + 1].x, d[2 * i + 1].y); o.w = pack2(d[2 * i + 1].z, d[2 * i + 1].w);
                    __builtin_amdgcn_raw_buffer_store_b128(o, myr, dof + i * 1024, 0, AUX_AGENT);
                }
            } else {
                const int ew = e - 64, p = ew >> 6, nt = (ew >> 2) & 15;
                const float* src = a.w + (size_t)(j * 64 + kq * 16) * NG + ew * 64 + n4 * 4;
                #pragma unroll
                for (int k = 0; k < 16; ++k) d[k] = *reinterpret_cast<const f32x4*>(src + (size_t)k * NG);
                #pragma unroll
                for (int i = 0; i < 4; ++i)
                #pragma unroll
                for (int c = 0; c < 2; ++c) {
                    u32x4 o;
                    o.x = pack2(d[c * 8 + 0][i], d[c * 8 + 1][i]); o.y = pack2(d[c * 8 + 2][i], d[c * 8 + 3][i]);
                    o.z = pack2(d[c * 8 + 4][i], d[c * 8 + 5][i]); o.w = pack2(d[c * 8 + 6][i], d[c * 8 + 7][i]);
                    *reinterpret_cast<u32x4*>(wl + lds_byte(n4 * 4 + i, kq * 16 + c * 8)) = o;
                }
                __builtin_amdgcn_wave_barrier();
                u32x4 t[8];
                #pragma unroll
                for (int i = 0; i < 8; ++i) t[i] = *reinterpret_cast<const u32x4*>(wl + i * 1024 + lane * 16);
                __builtin_amdgcn_wave_barrier();
                const unsigned dof = (unsigned)((p == a.rank) ? stg + MAT_B + (size_t)(kt * 16 + nt) * TILE_B
                                                              : OBOX_OFF + (size_t)((p * NGRP + j) * 16 + nt) * TILE_B)
                                     + (ew & 3) * 8192 + lane * 16;
                #pragma unroll
                for (int i = 0; i < 8; ++i) __builtin_amdgcn_raw_buffer_store_b128(t[i], myr, dof + i * 1024, 0, AUX_AGENT);
            }
            asm volatile("s_waitcnt vmcnt(0)" ::: "memory");
            __syncthreads();
            if (tid == 0) __hip_atomic_fetch_add(myf + F_CONV + 32 * j, 1u, __ATOMIC_RELAXED, __HIP_MEMORY_SCOPE_AGENT);
        }
    } else {
        const int p = (a.rank + xg) & (NR - 1);
        const int s = __builtin_amdgcn_readfirstlane(xi * 8 + wid);
        const __amdgpu_buffer_rsrc_t dr = mk_rsrc(a.ws[p], (unsigned)WS_NEED);
        unsigned* pcnt = reinterpret_cast<unsigned*>(a.ws[p]) + F_CNT;
        bool okc = true;
        u32x4 v[3][8];
        auto load_b = [&](int b, u32x4* d) {
            const int j = b / BPG, r = b % BPG, ti = s + CPX * 8 * (r >> 2);
            if (r == 0 && j < 3) {
                int o = 1;
                if (lane == 0)
                    for (int jj = j; jj < (j == 2 ? NGRP : j + 1); ++jj)
                        o &= wait_ge(myf + F_CONV + 32 * jj, call * 72, myf + F_TMO, call, j == 0) ? 1 : 0;
                asm volatile("s_waitcnt vmcnt(0)" ::: "memory");
                okc = okc && __builtin_amdgcn_readfirstlane(o);
            }
            const char* sp = (ti < 16) ? mys + (size_t)((j * 8 + a.rank) * 16 + ti) * TILE_B
                                       : obox + (size_t)((p * NGRP + j) * 16 + (ti - 16)) * TILE_B;
            sp += (b & 3) * 8192 + lane * 16;
            #pragma unroll
            for (int i = 0; i < 8; ++i) d[i] = *reinterpret_cast<const u32x4*>(sp + i * 1024);
        };
        auto store_b = [&](int b, const u32x4* d) {
            const int j = b / BPG, r = b % BPG, ti = s + CPX * 8 * (r >> 2);
            const unsigned dp = (unsigned)(stg + (ti < 16 ? 0 : MAT_B) + (size_t)((j * 8 + a.rank) * 16 + (ti & 15)) * TILE_B)
                                + (b & 3) * 8192 + lane * 16;
            #pragma unroll
            for (int i = 0; i < 8; ++i) __builtin_amdgcn_raw_buffer_store_b128(d[i], dr, dp + i * 1024, 0, AUX_SYSTEM);
        };
        auto signal = [&](int j) {
            if (lane == 0) {
                unsigned tk = __hip_atomic_fetch_add(wcnt + j, 1u, __ATOMIC_RELAXED, __HIP_MEMORY_SCOPE_WORKGROUP);
                if (tk == 7) __hip_atomic_fetch_add(pcnt + 32 * j, 1u, __ATOMIC_RELAXED, __HIP_MEMORY_SCOPE_SYSTEM);
            }
        };
        constexpr int NB = BPG * NGRP;
        load_b(0, v[0]); load_b(1, v[1]);
        #define COMM_STEP(T, BL, BS) do { const int t_ = (T); if (t_ < NB) { \
            if (t_ + 2 < NB) load_b(t_ + 2, v[BL]); \
            store_b(t_, v[BS]); \
            if (t_ % BPG == BPG - 1) asm volatile("s_barrier" ::: "memory"); \
            if (t_ % BPG == 0 && t_ > 0) { asm volatile("s_waitcnt vmcnt(16)" ::: "memory"); STAMP(1 + t_ / BPG); signal(t_ / BPG - 1); } \
        } } while (0)
        for (int t = 0; t < NB; t += 3) {
            COMM_STEP(t, 2, 0);
            COMM_STEP(t + 1, 0, 1);
            COMM_STEP(t + 2, 1, 2);
        }
        asm volatile("s_waitcnt vmcnt(0)" ::: "memory");
        STAMP(9);
        signal(NGRP - 1);
        const int oi = (xg - 1) * CPX + xi;
        if (tid == 0) {
            bool o = wait_ge(myf + F_OTIL + 32 * oi, call, myf + F_TMO, call, false);
            asm volatile("s_waitcnt vmcnt(0)" ::: "memory");
            wcnt[0] = o ? 1u : 0u;
        }
        __syncthreads();
        okc = okc && (*reinterpret_cast<volatile unsigned*>(wcnt) != 0);
        {
            const __amdgpu_buffer_rsrc_t orr = mk_rsrc(a.out[p], (unsigned)((size_t)OUT_ROWS * NG * 4));
            const char* sp = a.ws[a.rank] + OTIL_OFF + (size_t)oi * OTIL_B + lane * 16;
            const unsigned oo = (unsigned)((a.rank * NL + xi * 256) * 4) + lane * 16;
            const unsigned nb = __builtin_bit_cast(unsigned, __builtin_nanf(""));
            for (int r0 = wid * 32; r0 < wid * 32 + 32; r0 += 8) {
                u32x4 d[8];
                #pragma unroll
                for (int i = 0; i < 8; ++i) d[i] = *reinterpret_cast<const u32x4*>(sp + (r0 + i) * 1024);
                #pragma unroll
                for (int i = 0; i < 8; ++i) {
                    if (!okc) d[i] = u32x4{nb, nb, nb, nb};
                    __builtin_amdgcn_raw_buffer_store_b128(d[i], orr, oo + (unsigned)(r0 + i) * (unsigned)(NG * 4), 0, AUX_SYSTEM);
                }
            }
        }
        asm volatile("s_waitcnt vmcnt(0)" ::: "memory");
        __syncthreads();
        if (tid == 0) __hip_atomic_fetch_add(reinterpret_cast<unsigned*>(a.ws[p]) + F_DONE, 1u, __ATOMIC_RELAXED, __HIP_MEMORY_SCOPE_SYSTEM);
    }
    __syncthreads();

    STAMP(1);
    #define SA(b,h) (shm + ((b) * 2 + (h)) * 16384)
    #define SB(b,h) (shm + (4 + (b) * 2 + (h)) * 16384)
    #define STAGE(P, BASE, h, kt_) do { const char* _g = (BASE) + (size_t)(kt_) * (16 * TILE_B) + (h) * 16384 + tid * 16; \
        for (int _i = 0; _i < 2; ++_i) \
            __builtin_amdgcn_global_load_lds((const unsigned*)(_g + _i * 8192), (unsigned*)((P) + tid * 16 + _i * 8192), 16, 0, 0); } while (0)
    #define LDA(dst,b,h) for (int m = 0; m < 4; ++m) for (int k = 0; k < 2; ++k) \
        dst[m][k] = *reinterpret_cast<const half8*>(SA(b,h) + lds_byte(wr * 64 + m * 16 + fr, k * 32 + fq * 8))
    #define LDB(dst,b,h) for (int n = 0; n < 2; ++n) for (int k = 0; k < 2; ++k) \
        dst[n][k] = *reinterpret_cast<const half8*>(SB(b,h) + lds_byte(wc * 32 + n * 16 + fr, k * 32 + fq * 8))
    #define MMA(ai,bj,At,Bt) do { __builtin_amdgcn_s_setprio(1); \
        for (int m = 0; m < 4; ++m) for (int n = 0; n < 2; ++n) for (int k = 0; k < 2; ++k) \
            acc[ai][bj][m][n] = __builtin_amdgcn_mfma_f32_16x16x32_f16(Bt[n][k], At[m][k], acc[ai][bj][m][n], 0, 0, 0); \
        __builtin_amdgcn_s_setprio(0); } while (0)
    #define WAIT_V(n) asm volatile("s_waitcnt vmcnt(" #n ")" ::: "memory")
    #define WAIT_L(n) asm volatile("s_waitcnt lgkmcnt(" #n ")" ::: "memory")
    #define BAR __builtin_amdgcn_s_barrier()
    #define SCHED __builtin_amdgcn_sched_barrier(0)

    const int ntiles = (xg == 0) ? (xi < 7 * CPX ? 2 : 1) : (xi < CPX ? 0 : 1);
    const int wr = wid >> 2, wc = wid & 3, fr = lane & 15, fq = lane >> 4;
    bool ok = true;
    for (int tt = 0; tt < ntiles; ++tt) {
    const bool parked = (ntiles == 2 && tt == 0);
    const int txg = parked ? 1 + xi / CPX : xg, txi = parked ? xi % CPX : xi;
    const int q = (a.rank + txg) & (NR - 1);
    const int pm = 2 * q + (txi >> 4), pn = txi & 15;
    const char* gA = a.ws[a.rank] + stg + (size_t)pm * TILE_B;
    const char* gB = a.ws[a.rank] + stg + MAT_B + (size_t)pn * TILE_B;
    f32x4 acc[2][2][4][2] = {};
    half8 At[4][2], B0[2][2], B1[2][2];
    if (tt) __syncthreads();

    for (int g = 0; g < NGRP; ++g) {
        if (tt == 0) {
            if (tid == 0) {
                bool o = wait_ge(myf + F_CONV + 32 * g, call * 72, myf + F_TMO, call, false);
                o = wait_ge(myf + F_CNT + 32 * g, call * (7 * CPX), myf + F_TMO, call, g == 0) && o;
                asm volatile("s_waitcnt vmcnt(0)" ::: "memory");
                *reinterpret_cast<volatile int*>(shm) = o ? 1 : 0;
            }
            __syncthreads();
            ok = ok && (*reinterpret_cast<volatile int*>(shm) != 0);
            __syncthreads();
        }
        const int k0 = g * 8;
        STAMP(2 + g);
        STAGE(SB(0,0), gB, 0, k0); STAGE(SA(0,0), gA, 0, k0);
        STAGE(SB(0,1), gB, 1, k0); STAGE(SA(0,1), gA, 1, k0);
        if (wr == 1) BAR;
        WAIT_V(4); BAR;
        STAGE(SB(1,0), gB, 0, k0 + 1); STAGE(SA(1,0), gA, 0, k0 + 1); STAGE(SB(1,1), gB, 1, k0 + 1);
        WAIT_V(6); BAR;
        for (int t = k0; t < k0 + 6; t += 2) {
            LDB(B0,0,0); SCHED; LDA(At,0,0); STAGE(SA(1,1), gA, 1, t + 1);
            WAIT_L(8); BAR; WAIT_L(0); MMA(0,0,At,B0); BAR; SCHED;
            LDB(B1,0,1); STAGE(SB(0,0), gB, 0, t + 2);
            BAR; WAIT_L(0); MMA(0,1,At,B1); BAR;
            LDA(At,0,1); STAGE(SA(0,0), gA, 0, t + 2);
            BAR; WAIT_L(0); MMA(1,0,At,B0); BAR; SCHED;
            STAGE(SB(0,1), gB, 1, t + 2);
            WAIT_V(6); BAR; MMA(1,1,At,B1); BAR;
            LDB(B0,1,0); SCHED; LDA(At,1,0); STAGE(SA(0,1), gA, 1, t + 2);
            WAIT_L(8); BAR; WAIT_L(0); MMA(0,0,At,B0); BAR; SCHED;
            LDB(B1,1,1); STAGE(SB(1,0), gB, 0, t + 3);
            BAR; WAIT_L(0); MMA(0,1,At,B1); BAR;
            LDA(At,1,1); STAGE(SA(1,0), gA, 0, t + 3);
            BAR; WAIT_L(0); MMA(1,0,At,B0); BAR; SCHED;
            STAGE(SB(1,1), gB, 1, t + 3);
            WAIT_V(6); BAR; MMA(1,1,At,B1); BAR;
        }
        { LDB(B0,0,0); WAIT_V(0); LDA(At,0,0); STAGE(SA(1,1), gA, 1, k0 + 7);
          BAR; WAIT_L(0); MMA(0,0,At,B0); BAR;
          LDB(B1,0,1); BAR; WAIT_L(0); MMA(0,1,At,B1); BAR;
          LDA(At,0,1); WAIT_V(4); BAR; WAIT_L(0); MMA(1,0,At,B0); MMA(1,1,At,B1); BAR; }
        { LDB(B0,1,0); LDA(At,1,0); WAIT_V(2); BAR; WAIT_L(0); MMA(0,0,At,B0); BAR;
          LDB(B1,1,1); WAIT_V(0); BAR; WAIT_L(0); MMA(0,1,At,B1); BAR;
          LDA(At,1,1); BAR; WAIT_L(0); MMA(1,0,At,B0); MMA(1,1,At,B1); BAR; }
        if (wr == 0) BAR;
    }

    STAMP(10);
    {
        const int brow = pm * 256;
        const __amdgpu_buffer_rsrc_t orr = parked ? mk_rsrc(a.ws[a.rank], (unsigned)WS_NEED)
                                                  : mk_rsrc(a.out[q], (unsigned)((size_t)OUT_ROWS * NG * 4));
        const unsigned oo = (parked ? (unsigned)(OTIL_OFF + (size_t)xi * OTIL_B)
                                    : (unsigned)(((size_t)(brow - q * OUT_ROWS) * NG + a.rank * NL + pn * 256) * 4)) + lane * 16;
        const unsigned opitch = parked ? 1024u : (unsigned)(NG * 4);
        const float nanv = __builtin_nanf("");
        #pragma unroll
        for (int ai = 0; ai < 2; ++ai) {
            if (ai) __syncthreads();
            #pragma unroll
            for (int bj = 0; bj < 2; ++bj)
            #pragma unroll
            for (int m = 0; m < 4; ++m)
            #pragma unroll
            for (int n = 0; n < 2; ++n) {
                const int row = wr * 64 + m * 16 + fr, ch = bj * 32 + wc * 8 + n * 4 + fq;
                *reinterpret_cast<f32x4*>(shm + row * 1024 + ((ch ^ fr) << 4)) = acc[ai][bj][m][n];
            }
            __syncthreads();
            #pragma unroll 4
            for (int i = 0; i < 16; ++i) {
                const int row = wid * 16 + i;
                f32x4 v = *reinterpret_cast<const f32x4*>(shm + row * 1024 + ((lane ^ i) << 4)), r;
                #pragma unroll
                for (int e = 0; e < 4; ++e) {
                    float y = v[e];
                    float c = fminf(fmaxf(y, -60.0f), 60.0f);
                    r[e] = ok ? y / (1.0f + __expf(-c)) : nanv;
                }
                __builtin_amdgcn_raw_buffer_store_b128(__builtin_bit_cast(u32x4, r), orr, oo + (unsigned)(ai * 128 + row) * opitch, 0, AUX_SYSTEM);
            }
        }
        asm volatile("s_waitcnt vmcnt(0)" ::: "memory");
        __syncthreads();
        STAMP(11);
        if (tid == 0) {
            if (parked) __hip_atomic_store(myf + F_OTIL + 32 * xi, call, __ATOMIC_RELAXED, __HIP_MEMORY_SCOPE_AGENT);
            else __hip_atomic_fetch_add(reinterpret_cast<unsigned*>(a.ws[q]) + F_DONE, 1u, __ATOMIC_RELAXED, __HIP_MEMORY_SCOPE_SYSTEM);
        }
    }
    }
    if (tid == 0) {
        unsigned f = __hip_atomic_fetch_add(myf + F_FIN, 1u, __ATOMIC_RELAXED, __HIP_MEMORY_SCOPE_AGENT);
        if (f % NWG == NWG - 1) {
            if (!wait_ge(myf + F_DONE, call * NWG, myf + F_TMO, call, true))
                a.out[a.rank][(size_t)a.rank * NL] = __builtin_nanf("");
        }
    }
    STAMP(12);
}

extern "C" void kernel_launch(int rank, int nranks, ncclComm_t comm,
                              hipStream_t stream,
                              void* const* d_in, const int* in_sizes, int n_in,
                              void* d_out, int out_size,
                              const int* in_dtypes, int out_dtype,
                              void* const* d_out_all,
                              void* const* d_ws_all, size_t ws_bytes) {
    if (nranks != NR || n_in < 2 || ws_bytes < WS_NEED || out_size != OUT_ROWS * NG ||
        in_sizes[0] != MROWS * KS || in_sizes[1] != KS * NG) {
        fprintf(stderr, "gemm_rs: unexpected problem layout\n");
        return;
    }
    static bool attr_set = false;
    if (!attr_set) {
        HIP_CHECK(hipFuncSetAttribute((const void*)gemm_rs_silu, hipFuncAttributeMaxDynamicSharedMemorySize, SHM_B));
        attr_set = true;
    }
    Args a;
    a.x = (const float*)d_in[0];
    a.w = (const float*)d_in[1];
    for (int p = 0; p < NR; ++p) { a.out[p] = (float*)d_out_all[p]; a.ws[p] = (char*)d_ws_all[p]; }
    a.rank = rank;
    hipLaunchKernelGGL(gemm_rs_silu, dim3(NWG), dim3(NTHR), SHM_B, stream, a);
    HIP_CHECK(hipGetLastError());
}
```

```cpp
#include <hip/hip_runtime.h>
#include <rccl/rccl.h>
#include <cstdio>
#include <cstdint>

#define HIP_CHECK(expr) do { hipError_t _e = (expr); if (_e != hipSuccess) { \
    fprintf(stderr, "HIP error %s at %s:%d\n", hipGetErrorString(_e), __FILE__, __LINE__); \
    return; } } while (0)

typedef _Float16 half8 __attribute__((ext_vector_type(8)));
typedef float f32x4 __attribute__((ext_vector_type(4)));
typedef unsigned u32x4 __attribute__((ext_vector_type(4)));

constexpr int NR = 8;
constexpr int MROWS = 4096;
constexpr int KS = 512;
constexpr int NG = 32768;
constexpr int NL = 4096;
constexpr int OUT_ROWS = 512;
constexpr int NWG = 256;
constexpr int NTHR = 512;
constexpr int CPX = 4;
constexpr int NCOPY = 7 * CPX;
constexpr int NCONV = NWG - NCOPY;
constexpr int NGRP = 8;
constexpr size_t TILE_B = 32768;
constexpr size_t MAT_B = 16ull * 64 * TILE_B;
constexpr size_t STG_OFF = 1ull << 20;
constexpr size_t OBOX_OFF = STG_OFF + 4 * MAT_B;
constexpr size_t OTIL_OFF = OBOX_OFF + MAT_B;
constexpr size_t OTIL_B = 256 * 256 * 4;
constexpr size_t WS_NEED = OTIL_OFF + 256 * OTIL_B;
constexpr int F_TICKET = 0, F_FIN = 32, F_TMO = 64, F_DONE = 96;
constexpr int F_CNTA = 128, F_CNTB = 384, F_CONV = 640;
constexpr int F_OTIL = 1024;
constexpr int CONV0 = 30;
constexpr unsigned DONE_PER_CALL = 7 * CPX * 2 + 32;
constexpr unsigned long long DEADLINE_TICKS = 50000000ull;
constexpr int SHM_B = 131072;
constexpr int AUX_AGENT = 16, AUX_SYSTEM = 17;

#define STAMP(i) do {} while (0)

struct Args {
    const float* x;
    const float* w;
    float* out[NR];
    char* ws[NR];
    int rank;
};

__device__ __forceinline__ int lds_byte(int r, int c) {
    int st = (r >> 4) * 2 + (c >> 5), rr = r & 15, cc = c & 31, ob = rr * 64 + cc * 2;
    return st * 1024 + (ob ^ (((ob >> 9) & 1) << 5));
}
__device__ __forceinline__ void stage_rc(int b, int& R, int& C) {
    int st = b / 1024, sb = b % 1024, swz = sb ^ (((sb >> 9) & 1) << 5);
    R = (st >> 1) * 16 + swz / 64; C = (st & 1) * 32 + (swz % 64) / 2;
}

__device__ __forceinline__ bool wait_ge(unsigned* flag, unsigned target, unsigned* tmo, unsigned call, bool acq) {
    unsigned long long t0 = __builtin_amdgcn_s_memrealtime();
    bool ok = true;
    while (__hip_atomic_load(flag, __ATOMIC_RELAXED, __HIP_MEMORY_SCOPE_SYSTEM) < target) {
        if (__hip_atomic_load(tmo, __ATOMIC_RELAXED, __HIP_MEMORY_SCOPE_AGENT) == call ||
            __builtin_amdgcn_s_memrealtime() - t0 > DEADLINE_TICKS) {
            __hip_atomic_store(tmo, call, __ATOMIC_RELAXED, __HIP_MEMORY_SCOPE_AGENT);
            ok = false;
            break;
        }
        __builtin_amdgcn_s_sleep(16);
    }
    if (acq) __builtin_amdgcn_fence(__ATOMIC_ACQUIRE, "");
    else asm volatile("" ::: "memory");
    return ok;
}

__device__ __forceinline__ __amdgpu_buffer_rsrc_t mk_rsrc(const void* p, unsigned bytes) {
    const unsigned long long u = reinterpret_cast<unsigned long long>(p);
    const unsigned lo = __builtin_amdgcn_readfirstlane((unsigned)u), hi = __builtin_amdgcn_readfirstlane((unsigned)(u >> 32));
    return __builtin_amdgcn_make_buffer_rsrc(reinterpret_cast<void*>(((unsigned long long)hi << 32) | lo), 0, bytes, 0x00020000);
}

__device__ __forceinline__ unsigned pack2(float a, float b) {
    typedef _Float16 h2 __attribute__((ext_vector_type(2)));
    h2 v; v.x = (_Float16)a; v.y = (_Float16)b;
    return __builtin_bit_cast(unsigned, v);
}

__global__ __launch_bounds__(NTHR, 2) void gemm_rs_silu(Args a) {
    extern __shared__ __attribute__((aligned(16))) char shm[];
    const int tid = threadIdx.x;
    unsigned* const myf = reinterpret_cast<unsigned*>(a.ws[a.rank]);

    STAMP(0);
    if (tid == 0) {
        unsigned t = __hip_atomic_fetch_add(myf + F_TICKET, 1u, __ATOMIC_RELAXED, __HIP_MEMORY_SCOPE_AGENT);
        *reinterpret_cast<unsigned*>(shm) = t / NWG + 1;
    }
    __syncthreads();
    const unsigned call = __builtin_amdgcn_readfirstlane(*reinterpret_cast<volatile unsigned*>(shm));
    __syncthreads();
    const size_t stg = STG_OFF + (size_t)(call & 1) * 2 * MAT_B;

    const int xg = blockIdx.x & 7, xi = blockIdx.x >> 3;
    const bool is_copy = (xg != 0) && (xi < CPX);
    const int lane = tid & 63, wid = tid >> 6;
    char* const mys = a.ws[a.rank] + stg;
    char* const obox = a.ws[a.rank] + OBOX_OFF;
    unsigned* const wcnt = reinterpret_cast<unsigned*>(shm + 65536);
    if (tid < 32) wcnt[tid] = 0;
    __syncthreads();

    char* const wl = shm + wid * 8192;
    auto conv_load = [&](int j, int e, f32x4* d) {
        if (e < 64) {
            const float* src = a.x + (size_t)(e * 64) * KS + j * 64;
            #pragma unroll
            for (int i = 0; i < 8; ++i) {
                int R, C; stage_rc(i * 1024 + lane * 16, R, C);
                const f32x4* q = reinterpret_cast<const f32x4*>(src + R * KS + C);
                d[2 * i] = q[0]; d[2 * i + 1] = q[1];
            }
        } else {
            const float* src = a.w + (size_t)(j * 64 + (lane >> 4) * 16) * NG + (e - 64) * 64 + (lane & 15) * 4;
            #pragma unroll
            for (int k = 0; k < 16; ++k) d[k] = *reinterpret_cast<const f32x4*>(src + (size_t)k * NG);
        }
    };
    auto conv_store = [&](int e, const f32x4* d, __amdgpu_buffer_rsrc_t r, unsigned dof, bool sys) {
        u32x4 t[8];
        if (e < 64) {
            #pragma unroll
            for (int i = 0; i < 8; ++i) {
                t[i].x = pack2(d[2 * i].x, d[2 * i].y); t[i].y = pack2(d[2 * i].z, d[2 * i].w);
                t[i].z = pack2(d[2 * i + 1].x, d[2 * i + 1].y); t[i].w = pack2(d[2 * i + 1].z, d[2 * i + 1].w);
            }
        } else {
            #pragma unroll
            for (int i = 0; i < 4; ++i)
            #pragma unroll
            for (int c = 0; c < 2; ++c) {
                u32x4 o;
                o.x = pack2(d[c * 8 + 0][i], d[c * 8 + 1][i]); o.y = pack2(d[c * 8 + 2][i], d[c * 8 + 3][i]);
                o.z = pack2(d[c * 8 + 4][i], d[c * 8 + 5][i]); o.w = pack2(d[c * 8 + 6][i], d[c * 8 + 7][i]);
                *reinterpret_cast<u32x4*>(wl + lds_byte((lane & 15) * 4 + i, (lane >> 4) * 16 + c * 8)) = o;
            }
            __builtin_amdgcn_wave_barrier();
            #pragma unroll
            for (int i = 0; i < 8; ++i) t[i] = *reinterpret_cast<const u32x4*>(wl + i * 1024 + lane * 16);
            __builtin_amdgcn_wave_barrier();
        }
        if (sys) {
            #pragma unroll
            for (int i = 0; i < 8; ++i) __builtin_amdgcn_raw_buffer_store_b128(t[i], r, dof + i * 1024, 0, AUX_SYSTEM);
        } else {
            #pragma unroll
            for (int i = 0; i < 8; ++i) __builtin_amdgcn_raw_buffer_store_b128(t[i], r, dof + i * 1024, 0, AUX_AGENT);
        }
    };

    if (is_copy) {
        const int p = (a.rank + xg) & (NR - 1);
        const int s = __builtin_amdgcn_readfirstlane(xi * 8 + wid);
        const __amdgpu_buffer_rsrc_t dr = mk_rsrc(a.ws[p], (unsigned)WS_NEED);
        unsigned* const pf = reinterpret_cast<unsigned*>(a.ws[p]);
        bool okc = true;
        auto signal = [&](int slot, unsigned* ctr) {
            if (lane == 0) {
                unsigned tk = __hip_atomic_fetch_add(wcnt + slot, 1u, __ATOMIC_RELAXED, __HIP_MEMORY_SCOPE_WORKGROUP);
                if (tk == 7) __hip_atomic_fetch_add(ctr, 1u, __ATOMIC_RELAXED, __HIP_MEMORY_SCOPE_SYSTEM);
            }
        };
        {
            const unsigned ob = (unsigned)(stg + (size_t)(a.rank * 16) * TILE_B) + lane * 16;
            auto qe = [&](int i) { const int u = s + 32 * i; return u < 64 ? u : 64 + p * 64 + (u - 64); };
            auto qo = [&](int i) { const int u = s + 32 * i, h = u & 63;
                                   return ob + (u < 64 ? 0u : (unsigned)MAT_B) + (unsigned)(h >> 2) * (unsigned)TILE_B + (h & 3) * 8192; };
            f32x4 d0[16], d1[16];
            conv_load(0, qe(0), d0);
            conv_load(0, qe(1), d1);
            conv_store(qe(0), d0, dr, qo(0), true);
            conv_load(0, qe(2), d0);
            conv_store(qe(1), d1, dr, qo(1), true);
            if (s < 16) conv_load(0, qe(3), d1);
            conv_store(qe(2), d0, dr, qo(2), true);
            if (s < 16) conv_store(qe(3), d1, dr, qo(3), true);
            asm volatile("s_barrier" ::: "memory");
        }
        constexpr int NB1 = 7 * (NGRP - 1), NB = NB1 + NGRP;
        u32x4 v[3][4];
        auto load_b = [&](int b, u32x4* d) {
            if (b == 0 || b == 7) {
                int o = 1;
                if (lane == 0)
                    for (int jj = b ? 2 : 0; jj < (b ? NGRP : 2); ++jj)
                        o &= wait_ge(myf + F_CONV + 32 * jj, call * (jj ? 72 : CONV0), myf + F_TMO, call, jj == 0) ? 1 : 0;
                asm volatile("s_waitcnt vmcnt(0)" ::: "memory");
                okc = okc && __builtin_amdgcn_readfirstlane(o);
            }
            const int j = b < NB1 ? 1 + b / 7 : b - NB1;
            const int hb = b < NB1 ? s + 32 * (b % 7) : 224 + s, u = hb >> 1;
            const char* sp = (u < 64) ? mys + (size_t)((j * 8 + a.rank) * 16 + (u >> 2)) * TILE_B
                                      : obox + (size_t)((p * NGRP + j) * 16 + ((u - 64) >> 2)) * TILE_B;
            sp += (hb & 7) * 4096 + lane * 16;
            #pragma unroll
            for (int i = 0; i < 4; ++i) d[i] = *reinterpret_cast<const u32x4*>(sp + i * 1024);
        };
        auto store_b = [&](int b, const u32x4* d) {
            const int j = b < NB1 ? 1 + b / 7 : b - NB1;
            const int hb = b < NB1 ? s + 32 * (b % 7) : 224 + s, u = hb >> 1;
            const unsigned dp = (unsigned)(stg + (u < 64 ? 0 : MAT_B) + (size_t)((j * 8 + a.rank) * 16 + ((u & 63) >> 2)) * TILE_B)
                                + (hb & 7) * 4096 + lane * 16;
            #pragma unroll
            for (int i = 0; i < 4; ++i) __builtin_amdgcn_raw_buffer_store_b128(d[i], dr, dp + i * 1024, 0, AUX_SYSTEM);
        };
        auto signal_prev = [&](int b) {
            if (b <= NB1) signal(b / 7, pf + F_CNTA + 32 * (b / 7));
            else signal(8 + (b - 1 - NB1), pf + F_CNTB + 32 * (b - 1 - NB1));
        };
        load_b(0, v[0]); load_b(1, v[1]);
        #define COMM_STEP(T, BL, BS) do { const int t_ = (T); if (t_ < NB) { \
            if (t_ + 2 < NB) load_b(t_ + 2, v[BL]); \
            store_b(t_, v[BS]); \
            if (t_ >= NB1 || t_ % 7 == 6) asm volatile("s_barrier" ::: "memory"); \
            if (t_ > NB1 || t_ % 7 == 0) { \
                if (t_ + 2 < NB) asm volatile("s_waitcnt vmcnt(8)" ::: "memory"); \
                else asm volatile("s_waitcnt vmcnt(4)" ::: "memory"); \
                signal_prev(t_); } \
        } } while (0)
        for (int t = 0; t < NB; t += 3) {
            COMM_STEP(t, 2, 0);
            COMM_STEP(t + 1, 0, 1);
            COMM_STEP(t + 2, 1, 2);
        }
        asm volatile("s_waitcnt vmcnt(0)" ::: "memory");
        signal_prev(NB);
        STAMP(1);
        const __amdgpu_buffer_rsrc_t orr = mk_rsrc(a.out[p], (unsigned)((size_t)OUT_ROWS * NG * 4));
        const unsigned nb = __builtin_bit_cast(unsigned, __builtin_nanf(""));
        for (int i4 = 0; i4 < 4; ++i4) {
            const int uo = s + 32 * i4, qd = uo & 3;
            const int ti = (uo >> 2) - (i4 == 3 ? 24 : 0);
            const int pmi = i4 == 3 ? ti >> 2 : ti / 12, pn = i4 == 3 ? 12 + (ti & 3) : ti % 12;
            const int tile = (2 * p + pmi) * 16 + pn;
            int o = 1;
            if (lane == 0) o = wait_ge(myf + F_OTIL + 32 * (2 * tile + (qd >> 1)), call, myf + F_TMO, call, false) ? 1 : 0;
            asm volatile("s_waitcnt vmcnt(0)" ::: "memory");
            const bool okt = okc && __builtin_amdgcn_readfirstlane(o);
            if (i4 == 0) STAMP(2);
            if (i4 == 3) STAMP(4);
            const char* sp = a.ws[a.rank] + OTIL_OFF + (size_t)tile * OTIL_B + (size_t)qd * 65536 + lane * 16;
            const unsigned oo = (unsigned)(((size_t)(pmi * 256 + qd * 64) * NG + a.rank * NL + pn * 256) * 4) + lane * 16;
            u32x4 d0[8], d1[8];
            #pragma unroll
            for (int i = 0; i < 8; ++i) d0[i] = *reinterpret_cast<const u32x4*>(sp + i * 1024);
            for (int r0 = 0; r0 < 64; r0 += 16) {
                #pragma unroll
                for (int i = 0; i < 8; ++i) d1[i] = *reinterpret_cast<const u32x4*>(sp + (r0 + 8 + i) * 1024);
                #pragma unroll
                for (int i = 0; i < 8; ++i) {
                    if (!okt) d0[i] = u32x4{nb, nb, nb, nb};
                    __builtin_amdgcn_raw_buffer_store_b128(d0[i], orr, oo + (unsigned)(r0 + i) * (unsigned)(NG * 4), 0, AUX_SYSTEM);
                }
                if (r0 + 16 < 64) {
                    #pragma unroll
                    for (int i = 0; i < 8; ++i) d0[i] = *reinterpret_cast<const u32x4*>(sp + (r0 + 16 + i) * 1024);
                }
                #pragma unroll
                for (int i = 0; i < 8; ++i) {
                    if (!okt) d1[i] = u32x4{nb, nb, nb, nb};
                    __builtin_amdgcn_raw_buffer_store_b128(d1[i], orr, oo + (unsigned)(r0 + 8 + i) * (unsigned)(NG * 4), 0, AUX_SYSTEM);
                }
            }
            if (i4 >= 2) {
                asm volatile("s_waitcnt vmcnt(0)" ::: "memory");
                signal(14 + i4, pf + F_DONE);
                STAMP(i4 == 2 ? 3 : 5);
            }
        }
    } else {
        const int ci = (xg == 0) ? xi : 32 + (xg - 1) * (32 - CPX) + (xi - CPX);
        {
            const __amdgpu_buffer_rsrc_t myr = mk_rsrc(a.ws[a.rank], (unsigned)WS_NEED);
            for (int blk = ci; blk < CONV0 + 72 * (NGRP - 1); blk += NCONV) {
                const int j = blk < CONV0 ? 0 : 1 + (blk - CONV0) / 72;
                int bb;
                if (blk < 8) bb = blk;
                else if (blk < 16) bb = a.rank * 8 + blk;
                else if (blk < CONV0) { const int o2 = blk - 16, pp = (a.rank + 1 + (o2 >> 1)) & 7; bb = 8 + pp * 8 + 6 + (o2 & 1); }
                else bb = (blk - CONV0) % 72;
                const int e = bb * 8 + wid, kt = j * 8 + a.rank;
                f32x4 d[16];
                conv_load(j, e, d);
                unsigned dof;
                if (e < 64) dof = (unsigned)(stg + (size_t)(kt * 16 + (e >> 2)) * TILE_B);
                else {
                    const int ew = e - 64, p = ew >> 6, nt = (ew >> 2) & 15;
                    dof = (unsigned)((p == a.rank) ? stg + MAT_B + (size_t)(kt * 16 + nt) * TILE_B
                                                   : OBOX_OFF + (size_t)((p * NGRP + j) * 16 + nt) * TILE_B);
                }
                conv_store(e, d, myr, dof + (e & 3) * 8192 + lane * 16, false);
                asm volatile("s_waitcnt vmcnt(0)" ::: "memory");
                __syncthreads();
                if (tid == 0) __hip_atomic_fetch_add(myf + F_CONV + 32 * j, 1u, __ATOMIC_RELAXED, __HIP_MEMORY_SCOPE_AGENT);
            }
        }
        STAMP(1);

        #define SA(b,h) (shm + ((b) * 2 + (h)) * 16384)
        #define SB(b,h) (shm + (4 + (b) * 2 + (h)) * 16384)
        #define STAGE(P, BASE, h, kt_) do { const char* _g = (BASE) + (size_t)(kt_) * (16 * TILE_B) + (h) * 16384 + tid * 16; \
            for (int _i = 0; _i < 2; ++_i) \
                __builtin_amdgcn_global_load_lds((const unsigned*)(_g + _i * 8192), (unsigned*)((P) + tid * 16 + _i * 8192), 16, 0, 0); } while (0)
        #define LDA(dst,b,h) for (int m = 0; m < 4; ++m) for (int k = 0; k < 2; ++k) \
            dst[m][k] = *reinterpret_cast<const half8*>(SA(b,h) + lds_byte(wr * 64 + m * 16 + fr, k * 32 + fq * 8))
        #define LDB(dst,b,h) for (int n = 0; n < 2; ++n) for (int k = 0; k < 2; ++k) \
            dst[n][k] = *reinterpret_cast<const half8*>(SB(b,h) + lds_byte(wc * 32 + n * 16 + fr, k * 32 + fq * 8))
        #define MMA(ai,bj,At,Bt) do { __builtin_amdgcn_s_setprio(1); \
            for (int m = 0; m < 4; ++m) for (int n = 0; n < 2; ++n) for (int k = 0; k < 2; ++k) \
                acc[ai][bj][m][n] = __builtin_amdgcn_mfma_f32_16x16x32_f16(Bt[n][k], At[m][k], acc[ai][bj][m][n], 0, 0, 0); \
            __builtin_amdgcn_s_setprio(0); } while (0)
        #define WAIT_V(n) asm volatile("s_waitcnt vmcnt(" #n ")" ::: "memory")
        #define WAIT_L(n) asm volatile("s_waitcnt lgkmcnt(" #n ")" ::: "memory")
        #define BAR __builtin_amdgcn_s_barrier()
        #define SCHED __builtin_amdgcn_sched_barrier(0)

        const int ntiles = ci < 200 ? 1 : 2;
        const int wr = wid >> 2, wc = wid & 3, fr = lane & 15, fq = lane >> 4;
        bool ok = true;
        for (int tt = 0; tt < ntiles; ++tt) {
            const int H = ci < 164 ? 0 : ci < 200 ? 1 : tt;
            const int hi = ci < 164 ? ci : ci < 200 ? ci - 164 : tt ? ci - 164 : ci - 36;
            const int pm = hi & 15, pn = H * 12 + (hi >> 4), q = pm >> 1;
            const char* gA = mys + (size_t)pm * TILE_B;
            const char* gB = mys + MAT_B + (size_t)pn * TILE_B;
            f32x4 acc[2][2][4][2] = {};
            half8 At[4][2], B0[2][2], B1[2][2];

            for (int g = 0; g < NGRP; ++g) {
                if (tid == 0) {
                    bool o = wait_ge(myf + F_CONV + 32 * g, call * (g ? 72 : CONV0), myf + F_TMO, call, false);
                    o = wait_ge(myf + F_CNTA + 32 * g, call * NCOPY, myf + F_TMO, call, false) && o;
                    if (H) o = wait_ge(myf + F_CNTB + 32 * g, call * NCOPY, myf + F_TMO, call, false) && o;
                    if (g == 0 && tt == 0) __builtin_amdgcn_fence(__ATOMIC_ACQUIRE, "");
                    asm volatile("s_waitcnt vmcnt(0)" ::: "memory");
                    *reinterpret_cast<volatile int*>(shm) = o ? 1 : 0;
                }
                __syncthreads();
                ok = ok && (*reinterpret_cast<volatile int*>(shm) != 0);
                __syncthreads();
                const int k0 = g * 8;
                STAMP(2 + g);
                STAGE(SB(0,0), gB, 0, k0); STAGE(SA(0,0), gA, 0, k0);
                STAGE(SB(0,1), gB, 1, k0); STAGE(SA(0,1), gA, 1, k0);
                if (wr == 1) BAR;
                WAIT_V(4); BAR;
                STAGE(SB(1,0), gB, 0, k0 + 1); STAGE(SA(1,0), gA, 0, k0 + 1); STAGE(SB(1,1), gB, 1, k0 + 1);
                WAIT_V(6); BAR;
                for (int t = k0; t < k0 + 6; t += 2) {
                    LDB(B0,0,0); SCHED; LDA(At,0,0); STAGE(SA(1,1), gA, 1, t + 1);
                    WAIT_L(8); BAR; WAIT_L(0); MMA(0,0,At,B0); BAR; SCHED;
                    LDB(B1,0,1); STAGE(SB(0,0), gB, 0, t + 2);
                    BAR; WAIT_L(0); MMA(0,1,At,B1); BAR;
                    LDA(At,0,1); STAGE(SA(0,0), gA, 0, t + 2);
                    BAR; WAIT_L(0); MMA(1,0,At,B0); BAR; SCHED;
                    STAGE(SB(0,1), gB, 1, t + 2);
                    WAIT_V(6); BAR; MMA(1,1,At,B1); BAR;
                    LDB(B0,1,0); SCHED; LDA(At,1,0); STAGE(SA(0,1), gA, 1, t + 2);
                    WAIT_L(8); BAR; WAIT_L(0); MMA(0,0,At,B0); BAR; SCHED;
                    LDB(B1,1,1); STAGE(SB(1,0), gB, 0, t + 3);
                    BAR; WAIT_L(0); MMA(0,1,At,B1); BAR;
                    LDA(At,1,1); STAGE(SA(1,0), gA, 0, t + 3);
                    BAR; WAIT_L(0); MMA(1,0,At,B0); BAR; SCHED;
                    STAGE(SB(1,1), gB, 1, t + 3);
                    WAIT_V(6); BAR; MMA(1,1,At,B1); BAR;
                }
                { LDB(B0,0,0); WAIT_V(0); LDA(At,0,0); STAGE(SA(1,1), gA, 1, k0 + 7);
                  BAR; WAIT_L(0); MMA(0,0,At,B0); BAR;
                  LDB(B1,0,1); BAR; WAIT_L(0); MMA(0,1,At,B1); BAR;
                  LDA(At,0,1); WAIT_V(4); BAR; WAIT_L(0); MMA(1,0,At,B0); MMA(1,1,At,B1); BAR; }
                { LDB(B0,1,0); LDA(At,1,0); WAIT_V(2); BAR; WAIT_L(0); MMA(0,0,At,B0); BAR;
                  LDB(B1,1,1); WAIT_V(0); BAR; WAIT_L(0); MMA(0,1,At,B1); BAR;
                  LDA(At,1,1); BAR; WAIT_L(0); MMA(1,0,At,B0); MMA(1,1,At,B1); BAR; }
                if (wr == 0) BAR;
            }

            STAMP(10);
            const bool local = (q == a.rank);
            const __amdgpu_buffer_rsrc_t orr = local ? mk_rsrc(a.out[a.rank], (unsigned)((size_t)OUT_ROWS * NG * 4))
                                                     : mk_rsrc(a.ws[a.rank], (unsigned)WS_NEED);
            const unsigned oo = (local ? (unsigned)(((size_t)((pm & 1) * 256) * NG + a.rank * NL + pn * 256) * 4)
                                       : (unsigned)(OTIL_OFF + (size_t)(pm * 16 + pn) * OTIL_B)) + lane * 16;
            const unsigned opitch = local ? (unsigned)(NG * 4) : 1024u;
            const float nanv = __builtin_nanf("");
            #pragma unroll
            for (int ai = 0; ai < 2; ++ai) {
                if (ai) {
                    asm volatile("s_waitcnt vmcnt(0)" ::: "memory");
                    __syncthreads();
                    if (tid == 0 && !local)
                        __hip_atomic_store(myf + F_OTIL + 32 * (2 * (pm * 16 + pn)), call, __ATOMIC_RELAXED, __HIP_MEMORY_SCOPE_AGENT);
                }
                #pragma unroll
                for (int bj = 0; bj < 2; ++bj)
                #pragma unroll
                for (int m = 0; m < 4; ++m)
                #pragma unroll
                for (int n = 0; n < 2; ++n) {
                    const int row = wr * 64 + m * 16 + fr, ch = bj * 32 + wc * 8 + n * 4 + fq;
                    *reinterpret_cast<f32x4*>(shm + row * 1024 + ((ch ^ fr) << 4)) = acc[ai][bj][m][n];
                }
                __syncthreads();
                #pragma unroll 4
                for (int i = 0; i < 16; ++i) {
                    const int row = wid * 16 + i;
                    f32x4 v = *reinterpret_cast<const f32x4*>(shm + row * 1024 + ((lane ^ i) << 4)), r;
                    #pragma unroll
                    for (int e = 0; e < 4; ++e) {
                        float y = v[e];
                        float c = fminf(fmaxf(y, -60.0f), 60.0f);
                        r[e] = ok ? y / (1.0f + __expf(-c)) : nanv;
                    }
                    __builtin_amdgcn_raw_buffer_store_b128(__builtin_bit_cast(u32x4, r), orr, oo + (unsigned)(ai * 128 + row) * opitch, 0, AUX_SYSTEM);
                }
            }
            asm volatile("s_waitcnt vmcnt(0)" ::: "memory");
            __syncthreads();
            STAMP(11);
            if (tid == 0) {
                if (local) __hip_atomic_fetch_add(myf + F_DONE, 1u, __ATOMIC_RELAXED, __HIP_MEMORY_SCOPE_SYSTEM);
                else __hip_atomic_store(myf + F_OTIL + 32 * (2 * (pm * 16 + pn) + 1), call, __ATOMIC_RELAXED, __HIP_MEMORY_SCOPE_AGENT);
            }
        }
    }

    __syncthreads();
    if (tid == 0) {
        unsigned f = __hip_atomic_fetch_add(myf + F_FIN, 1u, __ATOMIC_RELAXED, __HIP_MEMORY_SCOPE_AGENT);
        if (f % NWG == NWG - 1) {
            if (!wait_ge(myf + F_DONE, call * DONE_PER_CALL, myf + F_TMO, call, true))
                a.out[a.rank][(size_t)a.rank * NL] = __builtin_nanf("");
        }
    }
    STAMP(12);
}

extern "C" void kernel_launch(int rank, int nranks, ncclComm_t comm,
                              hipStream_t stream,
                              void* const* d_in, const int* in_sizes, int n_in,
                              void* d_out, int out_size,
                              const int* in_dtypes, int out_dtype,
                              void* const* d_out_all,
                              void* const* d_ws_all, size_t ws_bytes) {
    if (nranks != NR || rank < 0 || rank >= NR || n_in < 2 || ws_bytes < WS_NEED || out_size != OUT_ROWS * NG ||
        in_sizes[0] != MROWS * KS || in_sizes[1] != KS * NG) {
        fprintf(stderr, "gemm_rs: unexpected problem layout\n");
        return;
    }
    static bool attr_set = false;
    if (!attr_set) {
        HIP_CHECK(hipFuncSetAttribute((const void*)gemm_rs_silu, hipFuncAttributeMaxDynamicSharedMemorySize, SHM_B));
        attr_set = true;
    }
    Args a;
    a.x = (const float*)d_in[0];
    a.w = (const float*)d_in[1];
    for (int p = 0; p < NR; ++p) { a.out[p] = (float*)d_out_all[p]; a.ws[p] = (char*)d_ws_all[p]; }
    a.rank = rank;
    hipLaunchKernelGGL(gemm_rs_silu, dim3(NWG), dim3(NTHR), SHM_B, stream, a);
    HIP_CHECK(hipGetLastError());
}
```
